```python
import numpy as np

kernel_name = "gpu_gemm_ar_m4096_k4096_n4096_bf16_qe4m3"


ABSMAX_RTOL = 0.09

COMPUTE_DTYPE = "bfloat16"
EPILOGUE = "quant_e4m3"
IN_QUANT = None
OUT_QUANT = "e4m3"
IN_DTYPES = {"x": "bfloat16", "w_mat": "bfloat16"}
WORLD_SIZE = 8

def _to_bf16(x):
    u = np.ascontiguousarray(x, dtype=np.float32).view(np.uint32)
    return ((u + 0x7FFF + ((u >> 16) & 1)) & 0xFFFF0000).view(np.float32)

def setup_inputs():

    x = _to_bf16(np.random.randn(4096, 4096).astype(np.float32))
    w_mat = _to_bf16((np.random.randn(4096, 4096) / np.sqrt(4096)).astype(np.float32))
    return {"x": x, "w_mat": w_mat}

def _snap_fp8(x, mant_bits, max_abs, min_normal):
    a = np.abs(x.astype(np.float64))
    _m, _e = np.frexp(a)
    q = np.where(a >= min_normal, np.ldexp(1.0, _e - 1 - mant_bits),
                 min_normal * 2.0 ** -mant_bits)
    snapped = np.minimum(np.round(a / q) * q, max_abs)
    return (np.sign(x) * snapped).astype(np.float32)
def _quant_dequant(y, amax):
    scale = amax / 448.0
    q = _snap_fp8(y / scale, 3, 448.0, 2.0 ** -6)
    return (q * scale).astype(np.float32)
def _epilogue(y):
    return _quant_dequant(y, np.abs(y).max())

def reference(x, w_mat):
    y = (x @ w_mat).astype(np.float32)
    return _epilogue(y)
```

```cpp
#include <hip/hip_runtime.h>
#include <rccl/rccl.h>
#include <cstdio>

#define HIP_CHECK(expr) do { hipError_t _e = (expr); if (_e != hipSuccess) { \
    fprintf(stderr, "HIP error %s at %s:%d\n", hipGetErrorString(_e), __FILE__, __LINE__); \
    return; } } while (0)

namespace {

typedef unsigned u32;
typedef unsigned long long u64;
typedef short bf16x8 __attribute__((ext_vector_type(8)));
typedef float f32x16 __attribute__((ext_vector_type(16)));
typedef float f32x2 __attribute__((ext_vector_type(2)));
typedef __bf16 bf16x2_t __attribute__((ext_vector_type(2)));
typedef unsigned u32x4 __attribute__((ext_vector_type(4)));

constexpr int NR = 8;
constexpr int MM = 4096, NN = 4096, KS = 512;
constexpr int UNIT = 8192;
#define CW_PER_DST 12
#define BURST 8
#define MIDDRAIN() do { if (BURST == 4 && dst != me) drain(); } while (0)
constexpr int NWG = 256;
constexpr int WPB = 12;
constexpr int NSLOT = 4;
constexpr int SLOT_BYTES = 4 * UNIT;
constexpr int LDS_ID = NSLOT * SLOT_BYTES;
constexpr int LDS_CTL = LDS_ID + UNIT;
constexpr int LDS_JUNK = LDS_CTL + 512;
constexpr int LDS_BYTES = LDS_JUNK + 256;
#define WARM_AHEAD 0
constexpr int C_CLAIM = 0  , C_SYNC = 3, C_AV = 4  , C_REL = 8  , C_TOUCH = 12  ,
              C_Q = 16  , C_TAIL = 96  ;
constexpr int SELF_WAVES = 128;
constexpr u64 DEADLINE = 50000000ull;

constexpr size_t OFF_XS = 0;
constexpr size_t OFF_WS = 8u << 20;
constexpr size_t OFF_PS = 16u << 20;
constexpr size_t OFF_LP = 20u << 20;
constexpr size_t OFF_FL = 24u << 20;
constexpr size_t FL_FX = 0;
constexpr size_t FL_FW = 4096;
constexpr size_t FL_FP = 8192;
constexpr size_t FL_LP = 12288;
constexpr size_t FL_FD = 16384;
constexpr size_t FL_L1 = 49152;
constexpr size_t FL_G = 53248;
constexpr size_t FL_TMO = 53504;
constexpr size_t FL_TK = 54272;
constexpr size_t FL_CNT = 57344;
constexpr size_t FL_DBG = 73728;
constexpr size_t FL_DBG2 = 327680;
constexpr size_t FL_DBG3 = 800000 / 8 * 8;
constexpr size_t FL_BYTES = 1u << 20;
constexpr size_t WS_NEED = OFF_FL + FL_BYTES;

struct Params {
    const unsigned short* x;
    const unsigned short* w;
    char* ws[NR];
    char* out[NR];
    int rank;
};

#define RLX __ATOMIC_RELAXED
#define SYS __HIP_MEMORY_SCOPE_SYSTEM
#define AGT __HIP_MEMORY_SCOPE_AGENT
#define DI __device__ __forceinline__
constexpr int AUX_WT = 17;

DI __amdgpu_buffer_rsrc_t rsrc_of(const void* p, u32 bytes) {
    return __builtin_amdgcn_make_buffer_rsrc((void*)p, 0, bytes, 0x00020000);
}
DI void ldsdrain() { asm volatile("" ::: "memory"); __builtin_amdgcn_s_waitcnt(0xC07F); asm volatile("" ::: "memory"); }
DI void drain() { asm volatile("" ::: "memory"); __builtin_amdgcn_s_waitcnt(0x0F70); asm volatile("" ::: "memory"); }

DI int row_quarter(int r, int u) { const int a = (r >> 1) & 1; return (r >> 2) == 0 ? 2 * a + u : a + 2 * u; }
DI int col_quarter(int r, int v) { const int b = r & 1; return (r >> 2) == 0 ? 2 * b + v : b + 2 * v; }
DI int row_of(int r, int rbi) {
    const int i = rbi & 15, kept = rbi >> 4, g = r >> 2;
    return row_quarter(r, i >> 3) * 1024 + (((kept ^ g) << 3) + (i & 7)) * 64;
}
DI int col_of(int r, int cb) { return col_quarter(r, cb >> 4) * 1024 + (cb & 15) * 64; }

struct Waiter {
    u32* tmo; u32 call; u64 t0; u32 n;
    DI void start() { t0 = __builtin_amdgcn_s_memrealtime(); n = 0; }
    DI bool give_up() {
        __builtin_amdgcn_s_sleep(1);
        if ((++n & 15u) != 0u) return false;
        if (__hip_atomic_load(tmo, RLX, AGT) == call) return true;
        if (__builtin_amdgcn_s_memrealtime() - t0 > DEADLINE) { __hip_atomic_store(tmo, call, RLX, AGT); return true; }
        return false;
    }
};

#define STAMP(slot) do { } while (0)
#define STAMP2(idx, val) do { } while (0)

DI float wave_max(float v) {
#pragma unroll
    for (int o = 32; o > 0; o >>= 1) v = fmaxf(v, __shfl_xor(v, o));
    return v;
}

DI void push_units(const Params& P, int me, int dst, int first, int nw, u32 cv, int lane, char* myws, int gw, Waiter& wt) {
    const bool same = (dst >> 2) == (me >> 2);
    const int rel = (me - dst) & 3;
    const int cbase = same ? (rel == 0 ? 0 : 17 + 5 * (rel - 1)) : 3 * rel;
    char* dws = P.ws[dst];
    const __amdgpu_buffer_rsrc_t drs = rsrc_of(dws, (u32)OFF_LP);
    const __amdgpu_buffer_rsrc_t lrs = rsrc_of(myws + OFF_LP, 4u << 20);
    u32* dfl = (u32*)(dws + OFF_FL);
    const u32* lfl = (const u32*)(myws + OFF_FL + FL_LP);
    u32* tk = (u32*)(myws + OFF_FL + FL_TK) + dst * 32;
    u32 call = 0u;
    const int rho = lane & 31, h = lane >> 5;
    const int pr = 16 * ((rho >> 2) & 1) + 4 * (rho >> 3) + (rho & 3);
    const int nch = same ? 5 : 3, j0 = same ? 0 : 5;
    const int sbase = ((dst >> 1) & 1) * 8 * 32 + (dst & 1) * 16;
    const int kbase = ((me >> 1) & 1) * 8 * 32 + (me & 1) * 16;
    u32* pflag = nullptr;
    auto draw = [&]() -> u32 { u32 v = 0u; if (lane == 0) v = __hip_atomic_fetch_add(tk, 1u, RLX, AGT); return v; };
    auto poff = [&](u32 nn) -> int { const int t = (int)min(max(nn, 192u), 319u) - 192; return ((t >> 2) & 7) * 32 + (t >> 5) * 4 + (t & 3); };
    u32 n = (u32)first;
    u32 peek = 0u;
#define AFTER_DRAIN() do { asm volatile("" : "+v"(cv)); call = __builtin_amdgcn_readfirstlane(cv) + 1u; wt.call = call; \
        nn = __builtin_amdgcn_readfirstlane(nx) - (call - 1u) * 320u + (u32)nw; \
        if (pflag != nullptr && lane == 0) __hip_atomic_store(pflag, call, RLX, SYS); \
        if (!same) peek = __hip_atomic_load(lfl + sbase + poff(nn), RLX, AGT); } while (0)
    while (n < 320u) {
        u32 nx, nn;
        if (n < 16u * nch || (n >= 48u * nch && n < 64u * nch)) {
            const int m = n < 16u * nch ? (int)n : (int)n - 48 * nch;
            const int j = j0 + (m >> 4), blk = (n < 16u * nch ? 0 : 16) + (m & 15), c = cbase + j;
            const unsigned short* src = P.x + (size_t)(row_of(dst, blk) + rho) * KS + j * 64 + 8 * h;
            u32x4 v[8];
#pragma unroll
            for (int f = 0; f < 8; ++f) v[f] = *(const u32x4*)(src + (size_t)(f >> 2) * 32 * KS + (f & 3) * 16);
            nx = draw();
            drain();
            AFTER_DRAIN();
            const u32 uoff = (u32)OFF_XS + (u32)(c * 32 + blk) * UNIT + lane * 16;
#pragma unroll
            for (int f = 0; f < 8; ++f) { if (f == 4) MIDDRAIN(); __builtin_amdgcn_raw_buffer_store_b128(v[f], drs, uoff + f * 1024, 0, AUX_WT); }
            pflag = dfl + FL_FX / 4 + blk * 32 + c;
        } else if (n < 48u * nch) {
            const int m = (int)n - 16 * nch, gam = m / (8 * nch), r = m - gam * 8 * nch;
            const int j = j0 + (r >> 3), blk = 16 * ((r >> 2) & 1) + 4 * gam + (r & 3), c = cbase + j;
            const unsigned short* src = P.w + (size_t)(j * 64 + 8 * h) * NN + col_of(dst, blk) + 2 * pr;
            u32 d[32];
#pragma unroll
            for (int e = 0; e < 32; ++e) d[e] = *(const u32*)(src + (size_t)((e >> 3) * 16 + (e & 7)) * NN);
            nx = draw();
            drain();
            AFTER_DRAIN();
            const u32 uoff = (u32)OFF_WS + (u32)(c * 32 + blk) * UNIT + lane * 16;
#pragma unroll
            for (int ks = 0; ks < 4; ++ks) {
                u32x4 t0, t1;
                if (ks == 2) MIDDRAIN();
#pragma unroll
                for (int e = 0; e < 4; ++e) {
                    t0[e] = __builtin_amdgcn_perm(d[ks * 8 + 2 * e + 1], d[ks * 8 + 2 * e], 0x05040100u);
                    t1[e] = __builtin_amdgcn_perm(d[ks * 8 + 2 * e + 1], d[ks * 8 + 2 * e], 0x07060302u);
                }
                __builtin_amdgcn_raw_buffer_store_b128(t0, drs, uoff + ks * 1024, 0, AUX_WT);
                __builtin_amdgcn_raw_buffer_store_b128(t1, drs, uoff + (4 + ks) * 1024, 0, AUX_WT);
            }
            pflag = dfl + FL_FW / 4 + blk * 32 + c;
        } else {
            const int off = poff(n);
            if (__builtin_amdgcn_readfirstlane(peek) < call) {
                drain();
                if (pflag != nullptr && lane == 0) __hip_atomic_store(pflag, call, RLX, SYS);
                pflag = nullptr;
                wt.start();
                while (__builtin_amdgcn_readfirstlane(__hip_atomic_load(lfl + sbase + off, RLX, AGT)) < call) { if (wt.give_up()) break; }
            }
            asm volatile("" ::: "memory");
            u32x4 v[8];
#pragma unroll
            for (int f = 0; f < 8; ++f) v[f] = __builtin_amdgcn_raw_buffer_load_b128(lrs, (u32)(sbase + off) * UNIT + f * 1024 + lane * 16, 0, 16);
            nx = draw();
            drain();
            AFTER_DRAIN();
            const u32 uoff = (u32)OFF_PS + (u32)(kbase + off) * UNIT + lane * 16;
#pragma unroll
            for (int f = 0; f < 8; ++f) { if (f == 4) MIDDRAIN(); __builtin_amdgcn_raw_buffer_store_b128(v[f], drs, uoff + f * 1024, 0, AUX_WT); }
            pflag = dfl + FL_FP / 4 + kbase + off;
        }
        n = nn;
    }
#undef AFTER_DRAIN
    drain();
    if (pflag != nullptr && lane == 0) __hip_atomic_store(pflag, call, RLX, SYS);
}

typedef __attribute__((address_space(3))) unsigned lds_u32;
#define WGP __HIP_MEMORY_SCOPE_WORKGROUP
DI u32 lds_ld(lds_u32* p) { return __builtin_amdgcn_readfirstlane(__hip_atomic_load(p, RLX, WGP)); }
DI void glds4k(const char* gsrc, u32 lds_dst) {
    u32 keep;
    asm volatile("s_mov_b32 %0, m0\n\ts_mov_b32 m0, %2\n\ts_nop 0\n\t"
                 "global_load_lds_dwordx4 %1, off\n\tglobal_load_lds_dwordx4 %1, off offset:1024\n\t"
                 "global_load_lds_dwordx4 %1, off offset:2048\n\tglobal_load_lds_dwordx4 %1, off offset:3072\n\t"
                 "s_mov_b32 m0, %0" : "=&s"(keep) : "v"(gsrc), "s"(lds_dst) : "memory");
}

__global__ void __launch_bounds__(64 * WPB, 3) gemm_ar_q(Params P) {
    __shared__ __attribute__((aligned(16))) unsigned char lds[LDS_BYTES];
    const int lane = threadIdx.x & 63;
    const int wv = __builtin_amdgcn_readfirstlane(threadIdx.x >> 6);
    const int me = P.rank;
    const int gb = blockIdx.x;
    const int xcd = gb & 7, grp = gb >> 3;
    const bool is_gemm = (grp & 1) == 0;
    const int ci = (grp >> 1) * 8 + xcd;
    if (!is_gemm && wv * 128 + ci >= 7 * CW_PER_DST && !(wv == WPB - 1 && ci == 127)) return;
    const int gw = gb * WPB + wv;
    char* const myws = P.ws[me];
    char* const myfl = myws + OFF_FL;
    lds_u32* const ctl = (lds_u32*)(lds + LDS_CTL);
    u32* cnt = (u32*)(myfl + FL_CNT) + gw;
    const u32 cv = __hip_atomic_load(cnt, RLX, AGT);
    Waiter wt; wt.tmo = (u32*)(myfl + FL_TMO);
    STAMP(0);
    if (!is_gemm && wv != WPB - 1) {
        push_units(P, me, (me + 1 + (wv * 128 + ci) % 7) & 7, (wv * 128 + ci) / 7, CW_PER_DST, cv, lane, myws, gw, wt);
        if (lane == 0) __hip_atomic_store(cnt, cv + 1u, RLX, AGT);
        STAMP(1);
        return;
    }
    const u32 call = __builtin_amdgcn_readfirstlane(cv) + 1u;
    if (lane == 0) __hip_atomic_store(cnt, call, RLX, AGT);
    wt.call = call;

    if (!is_gemm) {
        {
            const u64* l1 = (const u64*)(myfl + FL_L1);
            float mx;
            wt.start();
            for (;;) {
                bool ok = true; mx = 0.f;
#pragma unroll
                for (int i = 0; i < 8; ++i) {
                    const u64 gr = __hip_atomic_load(l1 + i * 64 + lane, RLX, AGT);
                    ok = ok && (u32)(gr >> 32) == call;
                    mx = fmaxf(mx, __uint_as_float((u32)gr));
                }
                if (__all(ok)) break;
                if (wt.give_up()) break;
            }
            mx = wave_max(mx);
            if (lane < NR) {
                u64* g = (u64*)(P.ws[lane] + OFF_FL + FL_G) + me;
                __hip_atomic_store(g, ((u64)call << 32) | __float_as_uint(mx), RLX, SYS);
            }
            STAMP(2);
        }
        return;
    }
    if (wv == 8) { ctl[lane] = 0u; ctl[64 + lane] = 0u; }
    if (wv < 8) {
        const int rho = lane & 31, h = lane >> 5, on = (wv >> 2) == ((wv >> 1) & 1), d = rho - 16 * (wv & 1) - 4 * h;
        u32x4 v;
#pragma unroll
        for (int e = 0; e < 4; ++e) {
            const int j0 = 8 * (e >> 1) + 2 * (e & 1);
            v[e] = on ? (d == j0 ? 0x3f80u : d == j0 + 1 ? 0x3f800000u : 0u) : 0u;
        }
        *(u32x4*)(lds + LDS_ID + wv * 1024 + lane * 16) = v;
    }
    __syncthreads();

    const int gi = grp >> 1;
    const int ip = (xcd >> 1) * 2 + (gi >> 3);
    const int jp = (gi & 7) * 2 + (xcd & 1);
    const int cb0 = jp * 2;
    const u32 lds0 = (u32)(size_t)(__attribute__((address_space(3))) unsigned char*)lds;

    if (wv >= 8) {
        const int slot = wv - 8;
        const u32* fx = (const u32*)(myfl + FL_FX) + ip * 2 * 32 + lane;
        const u32* fw = (const u32*)(myfl + FL_FW) + cb0 * 32 + lane;
        const u32* fpp = (const u32*)(myfl + FL_FP) + (ip * 2 + ((lane >> 1) & 1)) * 32 + cb0 + (lane & 1);
        const char* gx = myws + OFF_XS + (size_t)ip * 2 * UNIT + lane * 16;
        const char* gwp = myws + OFF_WS + (size_t)cb0 * UNIT + lane * 16;
        const char* gp = myws + OFF_PS + (size_t)(ip * 2 * 32 + cb0) * UNIT + lane * 16;
        const u32 l = lds0 + (u32)slot * SLOT_BYTES;
        u32 f0, f1, f2, f3, need = 0u;
#define POLL_ISSUE() do { f0 = __hip_atomic_load(fx, RLX, SYS); f1 = __hip_atomic_load(fx + 16 * 32, RLX, SYS); \
            f2 = __hip_atomic_load(fw, RLX, SYS); f3 = __hip_atomic_load(fpp, RLX, SYS); } while (0)
#define POLL_POST() do { const u64 b0 = __ballot(f0 >= call && f2 >= call), b1 = __ballot(f1 >= call && f2 >= call); \
            const u32 mine = lane == 0 ? (u32)b0 & (u32)(b0 >> 32) : lane == 1 ? (u32)b1 & (u32)(b1 >> 32) : (__all(f3 >= call) ? 1u : 0u); \
            if (lane < 3) __hip_atomic_fetch_or(ctl + C_AV + lane, mine, RLX, WGP); } while (0)
        wt.start();
        for (;;) {
            const u32 cw = __hip_atomic_load(ctl + (lane & 15), RLX, WGP);
            const u32 cl0 = __builtin_amdgcn_readlane(cw, C_CLAIM), cl1 = __builtin_amdgcn_readlane(cw, C_CLAIM + 1), clp = __builtin_amdgcn_readlane(cw, C_CLAIM + 2);
            if ((cl0 & cl1) == 0xffffffffu && clp != 0u) break;
            const u32 m0 = __builtin_amdgcn_readlane(cw, C_AV) & ~cl0, m1 = __builtin_amdgcn_readlane(cw, C_AV + 1) & ~cl1, mp = __builtin_amdgcn_readlane(cw, C_AV + 2) & ~clp;
            if ((m0 | m1 | mp) == 0u) {
                POLL_ISSUE();
                if (wt.give_up()) f0 = f1 = f2 = f3 = call;
                POLL_POST();
                continue;
            }
            if (__builtin_amdgcn_readlane(cw, C_REL + slot) != need) continue;
            const int ty = m0 != 0u ? 0 : mp != 0u ? 2 : 1;
            const int c = ty == 2 ? 0 : __builtin_ctz(ty ? m1 : m0);
            u32 old = 0u;
            if (lane == 0) old = __hip_atomic_fetch_or(ctl + C_CLAIM + ty, 1u << c, RLX, WGP);
            if ((__builtin_amdgcn_readfirstlane(old) >> c) & 1u) continue;
            const char* g0 = ty == 2 ? gp : gx + (size_t)(c * 32 + ty * 16) * UNIT;
            const char* g1 = ty == 2 ? gp + (size_t)32 * UNIT : gwp + (size_t)c * 32 * UNIT;
#pragma unroll
            for (int u = 0; u < 4; ++u) { glds4k(g0 + u * 4096, l + u * 4096); glds4k(g1 + u * 4096, l + 2 * UNIT + u * 4096); }
            const u32 t0 = __builtin_amdgcn_readlane(cw, C_TOUCH), t1 = __builtin_amdgcn_readlane(cw, C_TOUCH + 1);
            const u32 mm0 = ty == 0 ? m0 & ~(1u << c) : m0, mm1 = ty == 1 ? m1 & ~(1u << c) : m1;
            const int a0 = __builtin_popcount(mm0 & t0), a1 = a0 + __builtin_popcount(mm1 & t1);
            const bool w0 = (mm0 & ~t0) != 0u && a0 < WARM_AHEAD, w1 = (mm1 & ~t1) != 0u && a1 < WARM_AHEAD;
            const int tc = w0 ? __builtin_ctz(mm0 & ~t0) : w1 ? __builtin_ctz(mm1 & ~t1) : 0;
            if ((w0 || w1) && lane == 0) __hip_atomic_fetch_or(ctl + C_TOUCH + (w0 ? 0 : 1), 1u << tc, RLX, WGP);
            const char* h0 = (w0 || w1 ? gx + (size_t)(tc * 32 + (w0 ? 0 : 16)) * UNIT : g0) + lane * 112;
            const char* h1 = (w0 || w1 ? gwp + (size_t)tc * 32 * UNIT : g1) + lane * 112;
            u32 keep;
            asm volatile("s_mov_b32 %0, m0\n\ts_mov_b32 m0, %5\n\ts_nop 0\n\t"
                         "global_load_lds_dword %1, off\n\tglobal_load_lds_dword %2, off\n\t"
                         "global_load_lds_dword %3, off\n\tglobal_load_lds_dword %4, off\n\t"
                         "s_mov_b32 m0, %0" : "=&s"(keep) : "v"(h0), "v"(h0 + UNIT), "v"(h1), "v"(h1 + UNIT), "s"(lds0 + LDS_JUNK) : "memory");
            POLL_ISSUE();
            asm volatile("s_waitcnt vmcnt(8)" ::: "memory");
            if (lane == 0) {
                const u32 q = ty != 0, pos = __hip_atomic_fetch_add(ctl + C_TAIL + q, 1u, RLX, WGP);
                __hip_atomic_store(ctl + C_Q + q * 32 + pos, ((u32)slot + 1u) | (ty == 2 ? 256u : 0u) | ((u32)c << 9), RLX, WGP);
            }
            need += 4u;
            POLL_POST();
            wt.start();
        }
        return;
    }

    if (wv == 4) push_units(P, me, me, ci, SELF_WAVES, call - 1u, lane, myws, gw, wt);
    const int kept = wv >> 2, wr = (wv >> 1) & 1, wc = wv & 1;

    f32x16 acc[2][2];
#pragma unroll
    for (int a = 0; a < 2; ++a)
#pragma unroll
        for (int b = 0; b < 2; ++b)
#pragma unroll
            for (int e = 0; e < 16; ++e) acc[a][b][e] = 0.f;

    bf16x8 xa[4] = {}, wa[4] = {}, xb[4] = {}, wb[4] = {};
    const unsigned char* const lb = lds + lane * 16;
    lds_u32* const queue = ctl + C_Q + kept * 32;
    const int njobs = 32 + kept;
#define RDH(X, W, hf) do { _Pragma("unroll") for (int f = 0; f < 4; ++f) { const int o_ = ((f & 1) * 4 + 2 * (hf) + (f >> 1)) * 1024; \
            W[f] = *(const bf16x8*)(lb + ow + o_); X[f] = *(const bf16x8*)(lb + ox + o_); } \
        __builtin_amdgcn_sched_barrier(0); } while (0)
#define MMRD(X, W, NX, NW, hf, rel) do { _Pragma("unroll") for (int i = 0; i < 8; ++i) { \
            const int ks = i >> 2, mt = (i >> 1) & 1, jt = i & 1, f = i >> 1, o_ = ((f & 1) * 4 + 2 * (hf) + (f >> 1)) * 1024; \
            acc[mt][jt] = __builtin_amdgcn_mfma_f32_32x32x16_bf16(W[ks * 2 + jt], X[ks * 2 + mt], acc[mt][jt], 0, 0, 0); \
            if (i & 1) NX[f] = *(const bf16x8*)(lb + ox + o_); else NW[f] = *(const bf16x8*)(lb + ow + o_); \
            if (rel && i == 3) { asm volatile("" ::: "memory"); __builtin_amdgcn_s_waitcnt(0xC47F); \
                if (lane == 0) __hip_atomic_fetch_add(ctl + C_REL + cur, 1u, RLX, WGP); } \
            __builtin_amdgcn_sched_barrier(0); } } while (0)
#define JOBSTAMP(e) do { } while (0)
#define SETJOB(e) do { sl = (int)((e) & 255u) - 1; JOBSTAMP(e); \
        ox = sl * SLOT_BYTES + (((e) & 256u) ? (wv & 3) : wr) * UNIT; ow = ((e) & 256u) ? LDS_ID : sl * SLOT_BYTES + (2 + wc) * UNIT; } while (0)
    const int slot = (ip * 2 + wr) * 32 + cb0 + wc;
    {
        u32 e;
        int sl, ox, ow;
        while ((e = lds_ld(queue)) == 0u) __builtin_amdgcn_s_sleep(1);
        asm volatile("" ::: "memory");
        SETJOB(e);
        RDH(xa, wa, 0);
        for (int p = 1; p <= njobs; ++p) {
            const u32 en = __hip_atomic_load(queue + p, RLX, WGP);
            __builtin_amdgcn_sched_barrier(0);
            const int cur = sl;
            MMRD(xa, wa, xb, wb, 1, false);
            e = p < njobs ? __builtin_amdgcn_readfirstlane(en) : 0u;
            if (e != 0u) SETJOB(e);
            MMRD(xb, wb, xa, wa, 0, true);
            if (e == 0u && p < njobs) {
                while ((e = lds_ld(queue + p)) == 0u) __builtin_amdgcn_s_sleep(1);
                asm volatile("" ::: "memory");
                SETJOB(e);
                RDH(xa, wa, 0);
            }
        }
    }
    STAMP(1);

    if (!kept) {
        const __amdgpu_buffer_rsrc_t prs = rsrc_of(myws + OFF_LP, 4u << 20);
#pragma unroll
        for (int a = 0; a < 2; ++a)
#pragma unroll
            for (int b = 0; b < 2; ++b)
#pragma unroll
                for (int hh = 0; hh < 2; ++hh) {
                    u32x4 v;
#pragma unroll
                    for (int e = 0; e < 4; ++e) {
                        const f32x2 p2 = {acc[a][b][hh * 8 + 2 * e], acc[a][b][hh * 8 + 2 * e + 1]};
                        v[e] = __builtin_bit_cast(u32, __builtin_convertvector(p2, bf16x2_t));
                    }
                    __builtin_amdgcn_raw_buffer_store_b128(v, prs, (u32)slot * UNIT + ((a * 2 + b) * 2 + hh) * 1024 + lane * 16, 0, 16);
                }
        drain();
        if (lane == 0) __hip_atomic_store((u32*)(myfl + FL_LP) + slot, call, RLX, AGT);
        STAMP(2);
        while (lds_ld(ctl + C_SYNC) < 4u) __builtin_amdgcn_s_sleep(4);
        asm volatile("" ::: "memory");
    } else {
        float mx = 0.f;
#pragma unroll
        for (int a = 0; a < 2; ++a)
#pragma unroll
            for (int b = 0; b < 2; ++b)
#pragma unroll
                for (int e = 0; e < 16; ++e) mx = fmaxf(mx, fabsf(acc[a][b][e]));
        mx = wave_max(mx);
        if (lane == 0) __hip_atomic_store((u64*)(myfl + FL_L1) + slot, ((u64)call << 32) | __float_as_uint(mx), RLX, AGT);
        STAMP(2);
        const u64* g = (const u64*)(myfl + FL_G) + (lane & 7);
        float amax;
        wt.start();
        for (;;) {
            const u64 gr = __hip_atomic_load(g, RLX, SYS);
            amax = __uint_as_float((u32)gr);
            if (__all((u32)(gr >> 32) == call)) break;
            if (wt.give_up()) break;
        }
#pragma unroll
        for (int o = 4; o > 0; o >>= 1) amax = fmaxf(amax, __shfl_xor(amax, o));
        STAMP(3);
        while (lds_ld(ctl + C_REL) + lds_ld(ctl + C_REL + 1) + lds_ld(ctl + C_REL + 2) + lds_ld(ctl + C_REL + 3) < 4u * 65u) { }
        asm volatile("" ::: "memory");
        const float inv = amax > 0.f ? 448.0f / amax : 0.f;
        const int m = lane & 31, h = lane >> 5;
#pragma unroll
        for (int a = 0; a < 2; ++a) {
            u32 pk[8];
#pragma unroll
            for (int d = 0; d < 8; ++d) {
                int wd = __builtin_amdgcn_cvt_pk_fp8_f32(acc[a][0][2 * d] * inv, acc[a][1][2 * d] * inv, 0, false);
                wd = __builtin_amdgcn_cvt_pk_fp8_f32(acc[a][0][2 * d + 1] * inv, acc[a][1][2 * d + 1] * inv, wd, true);
                pk[d] = (u32)wd;
            }
            const int row = wr * 64 + a * 32 + m, ch = wc * 4 + h * 2;
            *(u32x4*)(lds + row * 128 + ((ch ^ (row & 7)) << 4)) = (u32x4){pk[0], pk[1], pk[2], pk[3]};
            *(u32x4*)(lds + row * 128 + (((ch + 1) ^ (row & 7)) << 4)) = (u32x4){pk[4], pk[5], pk[6], pk[7]};
        }
        asm volatile("s_waitcnt lgkmcnt(0)" ::: "memory");
        if (lane == 0) __hip_atomic_fetch_add(ctl + C_SYNC, 1u, RLX, WGP);
        while (lds_ld(ctl + C_SYNC) < 4u) { }
        asm volatile("" ::: "memory");
    }
    {
        const bool bad = __hip_atomic_load(wt.tmo, RLX, AGT) == call;
        const int p = (me + wv + xcd + gi) & 7;
        const __amdgpu_buffer_rsrc_t ors = rsrc_of(P.out[p], (u32)MM * NN);
        const int rl = (lane >> 3) * 16, ch = lane & 7;
        const u32 off0 = (u32)(row_of(me, 16 + ip * 2) + rl) * NN + col_of(me, cb0) + ch * 16;
        const int rot = (ci >> 1) + 2 * me;
#define AG_THR 1
        for (int s = 0; s < 16; s += AG_THR) {
            u32x4 v[AG_THR];
#pragma unroll
            for (int k = 0; k < AG_THR; ++k) {
                const int r = (s + k + rot) & 15;
                v[k] = *(const u32x4*)(lds + (rl + r) * 128 + ((ch ^ (r & 7)) << 4));
                if (bad) v[k] = (u32x4){0x7f7f7f7fu, 0x7f7f7f7fu, 0x7f7f7f7fu, 0x7f7f7f7fu};
            }
            drain();
#pragma unroll
            for (int k = 0; k < AG_THR; ++k) __builtin_amdgcn_raw_buffer_store_b128(v[k], ors, off0 + (u32)((s + k + rot) & 15) * NN, 0, AUX_WT);
        }
        drain();
        STAMP(4);
        if (lane == 0 && p != me) __hip_atomic_store((u32*)(P.ws[p] + OFF_FL + FL_FD) + me * 128 + ci, call, RLX, SYS);
        if (wv != me) {
            const u32* fd = (const u32*)(myfl + FL_FD) + wv * 128 + ci;
            wt.start();
            while (__builtin_amdgcn_readfirstlane(__hip_atomic_load(fd, RLX, SYS)) < call) {
                if (wt.give_up()) {
                    if (lane == 0) __builtin_amdgcn_raw_buffer_store_b32(0x7f7f7f7fu, rsrc_of(P.out[me], (u32)MM * NN), off0, 0, AUX_WT);
                    break;
                }
            }
        }
        STAMP(5);
    }
}

}

extern "C" void kernel_launch(int rank, int nranks, ncclComm_t comm,
                              hipStream_t stream,
                              void* const* d_in, const int* in_sizes, int n_in,
                              void* d_out, int out_size,
                              const int* in_dtypes, int out_dtype,
                              void* const* d_out_all,
                              void* const* d_ws_all, size_t ws_bytes) {
    if (nranks != NR || (unsigned)rank >= (unsigned)NR || n_in != 2 || ws_bytes < WS_NEED ||
        in_sizes[0] != MM * KS || in_sizes[1] != KS * NN || out_size != MM * NN) {
        fprintf(stderr, "kernel_launch: unexpected problem shape\n");
        return;
    }
    Params P;
    P.x = (const unsigned short*)d_in[0];
    P.w = (const unsigned short*)d_in[1];
    for (int p = 0; p < NR; ++p) { P.ws[p] = (char*)d_ws_all[p]; P.out[p] = (char*)d_out_all[p]; }
    P.rank = rank;
    gemm_ar_q<<<NWG, 64 * WPB, 0, stream>>>(P);
    HIP_CHECK(hipGetLastError());
}
```

```cpp
#include <hip/hip_runtime.h>
#include <rccl/rccl.h>
#include <cstdio>

#define HIP_CHECK(expr) do { hipError_t _e = (expr); if (_e != hipSuccess) { \
    fprintf(stderr, "HIP error %s at %s:%d\n", hipGetErrorString(_e), __FILE__, __LINE__); \
    return; } } while (0)

namespace {

typedef unsigned u32;
typedef unsigned long long u64;
typedef short bf16x8 __attribute__((ext_vector_type(8)));
typedef float f32x16 __attribute__((ext_vector_type(16)));
typedef float f32x2 __attribute__((ext_vector_type(2)));
typedef __bf16 bf16x2_t __attribute__((ext_vector_type(2)));
typedef unsigned u32x4 __attribute__((ext_vector_type(4)));

constexpr int NR = 8;
constexpr int MM = 4096, NN = 4096, KS = 512;
constexpr int UNIT = 8192;
#define CW_PER_DST 32
constexpr int NCOMM = 128;
constexpr u32 NPIECES = 1280u;
constexpr int NWG = 256;
constexpr int WPB = 12;
constexpr int NSLOT = 4;
constexpr int SLOT_BYTES = 4 * UNIT;
constexpr int LDS_ID = NSLOT * SLOT_BYTES;
constexpr int LDS_CTL = LDS_ID + UNIT;
constexpr int LDS_BYTES = LDS_CTL + 512;
constexpr int C_STATE = 0  ,
              C_REL = 8  , C_CLAIM = 24  , C_AV = 27  ,
              C_DONE = 32  , C_SYNC = 40  ;
constexpr u64 DEADLINE = 50000000ull;

constexpr size_t OFF_XS = 0;
constexpr size_t OFF_WS = 8u << 20;
constexpr size_t OFF_PS = 16u << 20;
constexpr size_t OFF_LP = 20u << 20;
constexpr size_t OFF_FL = 24u << 20;
constexpr size_t FL_FX = 0;
constexpr size_t FL_FW = 4096;
constexpr size_t FL_FP = 8192;
constexpr size_t FL_LP = 12288;
constexpr size_t FL_FD = 16384;
constexpr size_t FL_L1 = 49152;
constexpr size_t FL_G = 53248;
constexpr size_t FL_TMO = 53504;
constexpr size_t FL_TK = 54272;
constexpr size_t FL_CNT = 57344;
constexpr size_t FL_UC = 69632;
constexpr size_t FL_BYTES = 1u << 17;
constexpr size_t WS_NEED = OFF_FL + FL_BYTES;

__host__ __device__ constexpr u64 order_word(bool same, int i) {
    return same ? (i == 0 ? 0x2928842524802120ull : i == 1 ? 0x819031308c2d2c88ull : i == 2 ? 0x8a838682918d8985ull : i == 3 ? 0x4140938f8b87928eull : 0xd1d0cdcc49484544ull)
                : (i == 0 ? 0x2524030201002120ull : i == 1 ? 0x908292807060504ull : 0x4948454441400b0aull);
}

struct Params {
    const unsigned short* x;
    const unsigned short* w;
    char* ws[NR];
    char* out[NR];
    int rank;
};

#define RLX __ATOMIC_RELAXED
#define SYS __HIP_MEMORY_SCOPE_SYSTEM
#define AGT __HIP_MEMORY_SCOPE_AGENT
#define DI __device__ __forceinline__
constexpr int AUX_WT = 17;

DI __amdgpu_buffer_rsrc_t rsrc_of(const void* p, u32 bytes) {
    return __builtin_amdgcn_make_buffer_rsrc((void*)p, 0, bytes, 0x00020000);
}
DI void ldsdrain() { asm volatile("" ::: "memory"); __builtin_amdgcn_s_waitcnt(0xC07F); asm volatile("" ::: "memory"); }
DI void drain() { asm volatile("" ::: "memory"); __builtin_amdgcn_s_waitcnt(0x0F70); asm volatile("" ::: "memory"); }

DI int row_quarter(int r, int u) { const int a = (r >> 1) & 1; return (r >> 2) == 0 ? 2 * a + u : a + 2 * u; }
DI int col_quarter(int r, int v) { const int b = r & 1; return (r >> 2) == 0 ? 2 * b + v : b + 2 * v; }
DI int row_of(int r, int rbi) {
    const int i = rbi & 15, kept = rbi >> 4, g = r >> 2;
    return row_quarter(r, i >> 3) * 1024 + (((kept ^ g) << 3) + (i & 7)) * 64;
}
DI int col_of(int r, int cb) { return col_quarter(r, cb >> 4) * 1024 + (cb & 15) * 64; }

struct Waiter {
    u32* tmo; u32 call; u64 t0; u32 n;
    DI void start() { n = 0u; }
    DI bool give_up() {
        __builtin_amdgcn_s_sleep(1);
        if ((++n & 63u) != 0u) return false;
        if (__hip_atomic_load(tmo, RLX, AGT) == call) return true;
        const u64 now = __builtin_amdgcn_s_memrealtime();
        if (n == 64u) t0 = now;
        if (now - t0 > DEADLINE) { __hip_atomic_store(tmo, call, RLX, AGT); return true; }
        return false;
    }
};

DI float wave_max(float f) {
    u32 v = __float_as_uint(f);
    v = max(v, (u32)__builtin_amdgcn_update_dpp(0, (int)v, 0x111, 0xf, 0xf, true));
    v = max(v, (u32)__builtin_amdgcn_update_dpp(0, (int)v, 0x112, 0xf, 0xf, true));
    v = max(v, (u32)__builtin_amdgcn_update_dpp(0, (int)v, 0x114, 0xf, 0xf, true));
    v = max(v, (u32)__builtin_amdgcn_update_dpp(0, (int)v, 0x118, 0xf, 0xf, true));
    v = max(v, (u32)__builtin_amdgcn_update_dpp(0, (int)v, 0x142, 0xa, 0xf, true));
    v = max(v, (u32)__builtin_amdgcn_update_dpp(0, (int)v, 0x143, 0xc, 0xf, true));
    return __uint_as_float(__builtin_amdgcn_readlane(v, 63));
}

template <bool SELF>
DI void push_units(const Params& P, int me, int dst, int first, int nw, u32 cv, int lane, char* myws, Waiter& wt) {
    const bool same = (dst >> 2) == (me >> 2);
    const int rel = (me - dst) & 3;
    const int cbase = same ? (rel == 0 ? 0 : 17 + 5 * (rel - 1)) : 3 * rel;
    char* dws = P.ws[dst];
    const __amdgpu_buffer_rsrc_t drs = rsrc_of(dws, (u32)OFF_LP);
    const __amdgpu_buffer_rsrc_t lrs = rsrc_of(myws + OFF_LP, 4u << 20);
    u32* dfl = (u32*)(dws + OFF_FL);
    const u32* lfl = (const u32*)(myws + OFF_FL + FL_LP);
    u32* tk = (u32*)(myws + OFF_FL + FL_TK) + dst * 32;
    u32* uc = (u32*)(myws + OFF_FL + FL_UC) + dst * 320;
    u32 call = 0u;
    const int rho = lane & 31, h = lane >> 5;
    const int pr = 16 * ((rho >> 2) & 1) + 4 * (rho >> 3) + (rho & 3);
    const u32 nch = same ? 5 : 3, j0 = same ? 0 : 5;
    const int sbase = ((dst >> 1) & 1) * 8 * 32 + (dst & 1) * 16;
    const int kbase = ((me >> 1) & 1) * 8 * 32 + (me & 1) * 16;
    u32* pflag = nullptr;
    u32* pcnt = nullptr;
    auto draw = [&]() -> u32 { u32 v = 0u; if (!SELF && lane == 0) v = __hip_atomic_fetch_add(tk, 1u, RLX, AGT); return v; };
    auto count = [&]() -> u32 { u32 v = 0u; if (pcnt != nullptr && lane == 0) v = __hip_atomic_fetch_add(pcnt, 1u, RLX, AGT); return v; };
    auto flag_if_last = [&](u32 old) { if (lane == 0 && (old & 3u) == 3u) __hip_atomic_store(pflag, 4u * call, RLX, SYS); };
    auto flag_add = [&]() { if (lane == 0) __hip_atomic_fetch_add(pflag, 1u, RLX, SYS); };
    bool purg = false;
    u32 cold = 0u;
    u32* cflag = nullptr;
    u32 zero = (u32)lane; asm volatile("" : "+v"(zero));
    auto poff = [&](u32 nn) -> int { const int t = (int)min(max(nn, 192u), 319u) - 192; return ((t >> 2) & 7) * 32 + (t >> 5) * 4 + (t & 3); };
    u32 q = SELF ? (u32)first * 4u : (u32)first;
    u32 peek = 0u;
    while (q < NPIECES) {
        const u32 n = q >> 2, ks = q & 3u;
        u32x4 t0, t1;
        u32 uoff, nx;
        u32* nflag;
        const u32 ent = (u32)(order_word(same, (int)(n >> 6)) >> (8u * ((n >> 3) & 7u))) & 255u;
        const int j = (int)(j0 + ((ent >> 2) & 7u)), c = cbase + j, r = (int)(n & 7u);
        if (n < 64u * nch && (ent & 96u) != 0u) {
            const int blk = ((ent & 64u) ? 16 : 0) + 8 * (int)(ent & 1u) + r;
            const unsigned short* src = P.x + (size_t)(row_of(dst, blk) + rho) * KS + j * 64 + ks * 16 + 8 * h;
            t0 = *(const u32x4*)src;
            t1 = *(const u32x4*)(src + (size_t)32 * KS);
            nx = draw();
            drain();
            uoff = (u32)OFF_XS + (u32)(c * 32 + blk) * UNIT;
            nflag = dfl + FL_FX / 4 + blk * 32 + c;
        } else if (n < 64u * nch) {
            const int blk = 16 * (r >> 2) + 4 * (int)(ent & 3u) + (r & 3);
            const unsigned short* src = P.w + (size_t)(j * 64 + ks * 16 + 8 * h) * NN + col_of(dst, blk) + 2 * pr;
            u32 d[8];
#pragma unroll
            for (int e = 0; e < 8; ++e) d[e] = *(const u32*)(src + (size_t)e * NN);
            nx = draw();
            drain();
#pragma unroll
            for (int e = 0; e < 4; ++e) {
                t0[e] = __builtin_amdgcn_perm(d[2 * e + 1], d[2 * e], 0x05040100u);
                t1[e] = __builtin_amdgcn_perm(d[2 * e + 1], d[2 * e], 0x07060302u);
            }
            uoff = (u32)OFF_WS + (u32)(c * 32 + blk) * UNIT;
            nflag = dfl + FL_FW / 4 + blk * 32 + c;
        } else {
            const int off = poff(n);
            if (__builtin_amdgcn_readfirstlane(peek) < call) {
                drain();
                if (lane == 0 && (cold & 3u) == 3u) __hip_atomic_store(cflag, 4u * call, RLX, SYS);
                cold = 0u;
                if (purg) flag_add(); else flag_if_last(count());
                pcnt = nullptr; purg = false;
                wt.start();
                while (__builtin_amdgcn_readfirstlane(__hip_atomic_load(lfl + sbase + off, RLX, AGT)) < call) { if (wt.give_up()) break; }
            }
            asm volatile("" ::: "memory");
            t0 = __builtin_amdgcn_raw_buffer_load_b128(lrs, (u32)(sbase + off) * UNIT + ks * 1024 + lane * 16, 0, 16);
            t1 = __builtin_amdgcn_raw_buffer_load_b128(lrs, (u32)(sbase + off) * UNIT + (4 + ks) * 1024 + lane * 16, 0, 16);
            nx = draw();
            drain();
            uoff = (u32)OFF_PS + (u32)(kbase + off) * UNIT;
            nflag = dfl + FL_FP / 4 + kbase + off;
        }
        asm volatile("" : "+v"(cv));
        call = __builtin_amdgcn_readfirstlane(cv) + 1u; wt.call = call;
        const u32 qn = SELF ? (ks == 3u ? NPIECES : q + 1u) : __builtin_amdgcn_readfirstlane(nx) - (call - 1u) * NPIECES + (u32)nw;
        if (lane == 0 && (cold & 3u) == 3u) __hip_atomic_store(cflag, 4u * call, RLX, SYS);
        cold = 0u; cflag = pflag;
        if (purg) flag_add();
        asm volatile("" ::: "memory");
        if (pcnt != nullptr && lane == 0) cold = __hip_atomic_fetch_add(pcnt + zero, 1u, RLX, AGT);
        if (!SELF && !same && qn >= 768u) peek = __hip_atomic_load(lfl + sbase + poff(qn >> 2), RLX, AGT);
        __builtin_amdgcn_raw_buffer_store_b128(t0, drs, uoff + ks * 1024 + lane * 16, 0, AUX_WT);
        __builtin_amdgcn_raw_buffer_store_b128(t1, drs, uoff + (4 + ks) * 1024 + lane * 16, 0, AUX_WT);
        pflag = nflag;
        purg = !SELF && (n < 64u * nch ? (ent & 128u) != 0u : n >= 288u);
        pcnt = SELF || purg ? nullptr : uc + n;
        q = qn;
    }
    drain();
    if (lane == 0 && (cold & 3u) == 3u) __hip_atomic_store(cflag, 4u * call, RLX, SYS);
    if (purg) flag_add(); else flag_if_last(SELF ? 3u : count());
}

typedef __attribute__((address_space(3))) unsigned lds_u32;
typedef __attribute__((address_space(1))) unsigned g_u32;
typedef __attribute__((address_space(1))) unsigned long long g_u64;
#define WGP __HIP_MEMORY_SCOPE_WORKGROUP
DI u32 lds_ld(lds_u32* p) { return __builtin_amdgcn_readfirstlane(__hip_atomic_load(p, RLX, WGP)); }
DI void glds4k(const char* gsrc, u32 lds_dst) {
    u32 keep;
    asm volatile("s_mov_b32 %0, m0\n\ts_mov_b32 m0, %2\n\ts_nop 0\n\t"
                 "global_load_lds_dwordx4 %1, off sc1\n\tglobal_load_lds_dwordx4 %1, off offset:1024 sc1\n\t"
                 "global_load_lds_dwordx4 %1, off offset:2048 sc1\n\tglobal_load_lds_dwordx4 %1, off offset:3072 sc1\n\t"
                 "s_mov_b32 m0, %0" : "=&s"(keep) : "v"(gsrc), "s"(lds_dst) : "memory");
}

__global__ void __launch_bounds__(64 * WPB, 3) gemm_ar_q(Params P) {
    __shared__ __attribute__((aligned(16))) unsigned char lds[LDS_BYTES];
    const int lane = threadIdx.x & 63;
    const int wv = __builtin_amdgcn_readfirstlane(threadIdx.x >> 6);
    const int me = P.rank;
    const int gb = blockIdx.x;
    const int xcd = gb & 7, grp = gb >> 3;
    const bool is_gemm = (grp & 1) == 0;
    const int gi = grp >> 1;
    const int ci = gi * 8 + xcd;
    if (!is_gemm && wv * NCOMM + ci >= 7 * CW_PER_DST && !(wv == WPB - 1 && ci == NCOMM - 1)) return;
    const int gw = gb * WPB + wv;
    char* const myws = P.ws[me];
    char* const myfl = myws + OFF_FL;
    lds_u32* const ctl = (lds_u32*)(lds + LDS_CTL);
    u32* cnt = (u32*)(myfl + FL_CNT) + gw;
    const u32 cv = __hip_atomic_load(cnt, RLX, AGT);
    Waiter wt; wt.tmo = (u32*)(myfl + FL_TMO); wt.t0 = 0; wt.n = 0u;
    if (!is_gemm && wv != WPB - 1) {
        push_units<false>(P, me, (me + 1 + (wv * NCOMM + ci) % 7) & 7, (wv * NCOMM + ci) / 7, CW_PER_DST, cv, lane, myws, wt);
        if (lane == 0) __hip_atomic_store(cnt, cv + 1u, RLX, AGT);
        return;
    }
    const u32 call = __builtin_amdgcn_readfirstlane(cv) + 1u;
    if (lane == 0) __hip_atomic_store(cnt, call, RLX, AGT);
    wt.call = call;

    if (!is_gemm) {
        {
            const u64* l1 = (const u64*)(myfl + FL_L1);
            g_u64* g = (g_u64*)((u64*)(P.ws[lane & 7] + OFF_FL + FL_G) + me);
            asm volatile("" : "+v"(g));
            float mx;
            wt.start();
            for (;;) {
                bool ok = true; mx = 0.f;
#pragma unroll
                for (int i = 0; i < 8; ++i) {
                    const u64 gr = __hip_atomic_load(l1 + i * 64 + lane, RLX, AGT);
                    ok = ok && (u32)(gr >> 32) == call;
                    mx = __uint_as_float(max(__float_as_uint(mx), (u32)gr));
                }
                if (__all(ok)) break;
                if (wt.give_up()) break;
            }
            mx = wave_max(mx);
            if (lane < NR) __hip_atomic_store(g, ((u64)call << 32) | __float_as_uint(mx), RLX, SYS);
        }
        return;
    }
    if (wv == 8) { ctl[lane] = 0u; ctl[64 + lane] = 0u; }
    if (wv < 8) {
        const int rho = lane & 31, h = lane >> 5, on = (wv >> 2) == ((wv >> 1) & 1), d = rho - 16 * (wv & 1) - 4 * h;
        u32x4 v;
#pragma unroll
        for (int e = 0; e < 4; ++e) {
            const int j0 = 8 * (e >> 1) + 2 * (e & 1);
            v[e] = on ? (d == j0 ? 0x3f80u : d == j0 + 1 ? 0x3f800000u : 0u) : 0u;
        }
        *(u32x4*)(lds + LDS_ID + wv * 1024 + lane * 16) = v;
    }
    __syncthreads();

    const int ip = (xcd >> 1) * 2 + (gi >> 3);
    const int jp = (gi & 7) * 2 + (xcd & 1);
    const int cb0 = jp * 2;
    const u32 lds0 = (u32)(size_t)(__attribute__((address_space(3))) unsigned char*)lds;

    if (wv >= 8) {
        const int slot = wv - 8;
        const u32* fx = (const u32*)(myfl + FL_FX) + ip * 2 * 32 + lane;
        const u32* fw = (const u32*)(myfl + FL_FW) + cb0 * 32 + lane;
        const u32* fpp = (const u32*)(myfl + FL_FP) + (ip * 2 + ((lane >> 1) & 1)) * 32 + cb0 + (lane & 1);
        const char* gx = myws + OFF_XS + (size_t)ip * 2 * UNIT + lane * 16;
        const char* gwp = myws + OFF_WS + (size_t)cb0 * UNIT + lane * 16;
        const char* gp = myws + OFF_PS + (size_t)(ip * 2 * 32 + cb0) * UNIT + lane * 16;
        const u32 l = lds0 + (u32)slot * SLOT_BYTES;
        const u32 own = 0x11111111u << slot, call4 = 4u * call;
        u32 f0, f1, f2, f3, fills = 0u;
#define POLL_ISSUE() do { f0 = __hip_atomic_load(fx, RLX, SYS); f1 = __hip_atomic_load(fx + 16 * 32, RLX, SYS); \
            f2 = __hip_atomic_load(fw, RLX, SYS); f3 = __hip_atomic_load(fpp, RLX, SYS); } while (0)
#define POLL_POST() do { const u64 b0 = __ballot(f0 >= call4 && f2 >= call4), b1 = __ballot(f1 >= call4 && f2 >= call4); \
            const u32 mine = lane == 0 ? (u32)b0 & (u32)(b0 >> 32) : lane == 1 ? (u32)b1 & (u32)(b1 >> 32) : (__all(f3 >= call4) ? 1u : 0u); \
            if (lane < 3) __hip_atomic_fetch_or(ctl + C_AV + lane, mine, RLX, WGP); } while (0)
        wt.start();
        for (;;) {
            const u32 cw = __hip_atomic_load(ctl + (lane & 31), RLX, WGP);
            const u32 cl0 = __builtin_amdgcn_readlane(cw, C_CLAIM), cl1 = __builtin_amdgcn_readlane(cw, C_CLAIM + 1), clp = __builtin_amdgcn_readlane(cw, C_CLAIM + 2);
            if ((cl0 & cl1) == 0xffffffffu && clp != 0u) break;
            const u32 m0 = __builtin_amdgcn_readlane(cw, C_AV) & ~cl0, m1 = __builtin_amdgcn_readlane(cw, C_AV + 1) & ~cl1, mp = __builtin_amdgcn_readlane(cw, C_AV + 2) & ~clp;
            if ((m0 | m1 | mp) == 0u) {
                if (slot >= 2) { __builtin_amdgcn_s_sleep(1); continue; }
                POLL_ISSUE();
                if (wt.give_up()) f0 = f1 = f2 = f3 = call4;
                POLL_POST();
                continue;
            }
            const bool free_now = (((u32)__ballot(cw == fills) >> (C_REL + slot * 4)) & 15u) == 15u;
            if (m0 == 0u && !free_now) continue;
            const int ty = m0 != 0u ? 0 : mp != 0u ? 2 : 1;
            const u32 m = ty == 0 ? m0 : ty == 1 ? m1 : 1u;
            const int c = __builtin_ctz((m & own) != 0u ? m & own : m);
            u32 old = 0u;
            if (lane == 0) old = __hip_atomic_fetch_or(ctl + C_CLAIM + (ty == 2 ? 2 : ty), 1u << c, RLX, WGP);
            if ((__builtin_amdgcn_readfirstlane(old) >> c) & 1u) continue;
            if (!free_now) while (!__all(__hip_atomic_load(ctl + C_REL + slot * 4 + (lane & 3), RLX, WGP) == fills)) { }
            const char* g0 = ty == 2 ? gp : gx + (size_t)(c * 32 + ty * 16) * UNIT;
            const char* g1 = ty == 2 ? gp + (size_t)32 * UNIT : gwp + (size_t)c * 32 * UNIT;
#pragma unroll
            for (int u = 0; u < 4; ++u) { glds4k(g0 + u * 4096, l + u * 4096); glds4k(g1 + u * 4096, l + 2 * UNIT + u * 4096); }
            POLL_ISSUE();
            asm volatile("s_waitcnt vmcnt(4)" ::: "memory");
            ++fills;
            if (lane == 0) __hip_atomic_store(ctl + C_STATE + (ty != 0 ? 4 : 0) + slot, (fills << 16) | ((u32)c << 9) | (ty == 2 ? 256u : 0u), RLX, WGP);
            POLL_POST();
            wt.start();
        }
        return;
    }

    if (wv >= 4 && (wv - 4) * 128 + ci < 320) push_units<true>(P, me, me, (wv - 4) * 128 + ci, 0, call - 1u, lane, myws, wt);
    const int kept = wv >> 2, wr = (wv >> 1) & 1, wc = wv & 1;

    f32x16 acc[2][2];
#pragma unroll
    for (int a = 0; a < 2; ++a)
#pragma unroll
        for (int b = 0; b < 2; ++b)
#pragma unroll
            for (int e = 0; e < 16; ++e) acc[a][b][e] = 0.f;

    bf16x8 xa[4] = {}, wa[4] = {}, xb[4] = {}, wb[4] = {};
    const unsigned char* const lb = lds + lane * 16;
    const int njobs = 32 + kept;
#define RDH(X, W, hf) do { _Pragma("unroll") for (int f = 0; f < 4; ++f) { const int o_ = ((f & 1) * 4 + 2 * (hf) + (f >> 1)) * 1024; \
            W[f] = *(const bf16x8*)(lb + ow + o_); X[f] = *(const bf16x8*)(lb + ox + o_); } \
        __builtin_amdgcn_sched_barrier(0); } while (0)
#define MMRD(X, W, NX, NW, hf, rel) do { _Pragma("unroll") for (int i = 0; i < 8; ++i) { \
            const int ks = i >> 2, mt = (i >> 1) & 1, jt = i & 1, f = i >> 1, o_ = ((f & 1) * 4 + 2 * (hf) + (f >> 1)) * 1024; \
            acc[mt][jt] = __builtin_amdgcn_mfma_f32_32x32x16_bf16(W[ks * 2 + jt], X[ks * 2 + mt], acc[mt][jt], 0, 0, 0); \
            if (i & 1) NX[f] = *(const bf16x8*)(lb + ox + o_); else NW[f] = *(const bf16x8*)(lb + ow + o_); \
            if (rel && i == 3) { asm volatile("" ::: "memory"); __builtin_amdgcn_s_waitcnt(0xC47F); \
                if (lane == 0) __hip_atomic_store(ctl + C_REL + cur * 4 + (wv & 3), curfill, RLX, WGP); } \
            __builtin_amdgcn_sched_barrier(0); } } while (0)
#define PICK1(st) do { asm volatile("" : "+v"(st)); pm = (u32)__ballot(((st) >> 16) != seen) & 15u; asm volatile("" : "+s"(pm)); } while (0)
#define PICK2() do { const int n_ = (sl + 1 + __builtin_ctz(((pm * 17u) | 256u) >> (sl + 1))) & 3; sl = pm ? n_ : sl; asm volatile("" : "+s"(sl)); } while (0)
#define PICK3(st) do { e = __builtin_amdgcn_readlane(st, sl); e = pm ? e : 0u; asm volatile("" : "+s"(e)); } while (0)
#define PICK4() do { if (pm != 0u && (lane & 3) == sl) seen = e >> 16; \
        ox = sl * SLOT_BYTES + ((e & 256u) ? (wv & 3) : wr) * UNIT; ow = (e & 256u) ? LDS_ID : sl * SLOT_BYTES + (2 + wc) * UNIT; \
        asm volatile("" : "+s"(ox), "+s"(ow), "+v"(seen)); } while (0)
#define WAITJOB(nap) do { for (;;) { u32 st = __hip_atomic_load(states, RLX, WGP); PICK1(st); \
            if (pm != 0u) { PICK2(); PICK3(st); PICK4(); break; } nap; } asm volatile("" ::: "memory"); } while (0)
    const int slot = (ip * 2 + wr) * 32 + cb0 + wc;
    {
        u32 e = 0u, pm, seen = 0u;
        int sl = 3, ox, ow;
        lds_u32* const states = ctl + C_STATE + kept * 4 + (lane & 3);
        WAITJOB(__builtin_amdgcn_s_sleep(1));
        RDH(xa, wa, 0);
        for (int p = 1; p <= njobs; ++p) {
            u32 en = __hip_atomic_load(states, RLX, WGP);
            const int cur = sl;
            const u32 curfill = e >> 16;
            const unsigned char* const px = lb + ox;
            const unsigned char* const pw = lb + ow;
            __builtin_amdgcn_sched_barrier(0);
#pragma unroll
            for (int i = 0; i < 8; ++i) {
                const int ks = i >> 2, mt = (i >> 1) & 1, jt = i & 1, f = i >> 1, o_ = ((f & 1) * 4 + 2 + (f >> 1)) * 1024;
                acc[mt][jt] = __builtin_amdgcn_mfma_f32_32x32x16_bf16(wa[ks * 2 + jt], xa[ks * 2 + mt], acc[mt][jt], 0, 0, 0);
                if (i & 1) xb[f] = *(const bf16x8*)(px + o_); else wb[f] = *(const bf16x8*)(pw + o_);
                if (i == 3) PICK1(en);
                if (i == 4) PICK2();
                if (i == 5) PICK3(en);
                if (i == 6) PICK4();
                __builtin_amdgcn_sched_barrier(0);
            }
            MMRD(xb, wb, xa, wa, 0, true);
            if (e == 0u && p < njobs) {
                WAITJOB((void)0);
                RDH(xa, wa, 0);
            }
        }
        ldsdrain();
    }
    if (lane == 0) __hip_atomic_store(ctl + C_DONE + wv, 1u, RLX, WGP);

    const int p = (me + wv + xcd + gi) & 7;
    const __amdgpu_buffer_rsrc_t ors = rsrc_of(P.out[p], (u32)MM * NN);
    g_u32* fdp = (g_u32*)((u32*)(P.ws[p] + OFF_FL + FL_FD) + me * 128 + ci);
    const int rl = (lane >> 3) * 16, ch = lane & 7, rot = (ci >> 1) + 2 * me;
    u32 off0 = (u32)(row_of(me, 16 + ip * 2) + rl) * NN + col_of(me, cb0) + ch * 16;
    asm volatile("" : "+v"(off0), "+s"(fdp) : "s"(ors));

    if (!kept) {
        const __amdgpu_buffer_rsrc_t prs = rsrc_of(myws + OFF_LP, 4u << 20);
#pragma unroll
        for (int f = 0; f < 8; ++f) {
            const int a = f >> 2, b = (f >> 1) & 1, hh = f & 1;
            u32x4 v;
#pragma unroll
            for (int e = 0; e < 4; ++e) {
                const f32x2 p2 = {acc[a][b][hh * 8 + 2 * e], acc[a][b][hh * 8 + 2 * e + 1]};
                v[e] = __builtin_bit_cast(u32, __builtin_convertvector(p2, bf16x2_t));
            }
            __builtin_amdgcn_raw_buffer_store_b128(v, prs, (u32)slot * UNIT + f * 1024 + lane * 16, 0, AUX_WT);
        }
        drain();
        if (lane == 0) __hip_atomic_store((u32*)(myfl + FL_LP) + slot, call, RLX, SYS);
        while (!__all(__hip_atomic_load(ctl + C_DONE + 4 + (lane & 3), RLX, WGP) != 0u)) __builtin_amdgcn_s_sleep(8);
        while (!__all(__hip_atomic_load(ctl + C_SYNC + (lane & 3), RLX, WGP) != 0u)) __builtin_amdgcn_s_sleep(1);
        asm volatile("" ::: "memory");
    } else {
        u32 mx = 0u;
#pragma unroll
        for (int a = 0; a < 2; ++a)
#pragma unroll
            for (int b = 0; b < 2; ++b)
#pragma unroll
                for (int e = 0; e < 16; ++e) mx = max(mx, __float_as_uint(acc[a][b][e]) & 0x7fffffffu);
        mx = __float_as_uint(wave_max(__uint_as_float(mx)));
        if (lane == 0) __hip_atomic_store((u64*)(myfl + FL_L1) + slot, ((u64)call << 32) | mx, RLX, AGT);
        while (!__all(__hip_atomic_load(ctl + C_DONE + (lane & 7), RLX, WGP) != 0u)) { }
        const u64* g = (const u64*)(myfl + FL_G) + (lane & 7);
        float amax;
        wt.start();
        for (;;) {
            const u64 gr = __hip_atomic_load(g, RLX, SYS);
            amax = __uint_as_float((u32)gr);
            if (__all((u32)(gr >> 32) == call)) break;
            if (wt.give_up()) break;
        }
        const u32 tm = __hip_atomic_load(wt.tmo, RLX, AGT);
        amax = wave_max(amax);
        const float inv = amax == 0.f ? 0.f : 448.0f / amax;
        const int m = lane & 31, h = lane >> 5;
        u32 pk[2][8];
#pragma unroll
        for (int a = 0; a < 2; ++a)
#pragma unroll
            for (int d = 0; d < 8; ++d) {
                int wd = __builtin_amdgcn_cvt_pk_fp8_f32(acc[a][0][2 * d] * inv, acc[a][1][2 * d] * inv, 0, false);
                wd = __builtin_amdgcn_cvt_pk_fp8_f32(acc[a][0][2 * d + 1] * inv, acc[a][1][2 * d + 1] * inv, wd, true);
                pk[a][d] = (u32)wd;
            }
        __builtin_amdgcn_sched_barrier(0);
        const u32 nanm = tm == call ? 0x7f7f7f7fu : 0u;
#pragma unroll
        for (int a = 0; a < 2; ++a) {
            const int row = wr * 64 + a * 32 + m, c2 = wc * 4 + h * 2;
            *(u32x4*)(lds + row * 128 + ((c2 ^ (row & 7)) << 4)) = (u32x4){pk[a][0] | nanm, pk[a][1] | nanm, pk[a][2] | nanm, pk[a][3] | nanm};
            *(u32x4*)(lds + row * 128 + (((c2 + 1) ^ (row & 7)) << 4)) = (u32x4){pk[a][4] | nanm, pk[a][5] | nanm, pk[a][6] | nanm, pk[a][7] | nanm};
        }
        ldsdrain();
        if (lane == 0) __hip_atomic_store(ctl + C_SYNC + (wv & 3), 1u, RLX, WGP);
        while (!__all(__hip_atomic_load(ctl + C_SYNC + (lane & 3), RLX, WGP) != 0u)) { }
        asm volatile("" ::: "memory");
    }
    for (int s = 0; s < 16; ++s) {
        const int r = (s + rot) & 15;
        const u32x4 v = *(const u32x4*)(lds + (rl + r) * 128 + ((ch ^ (r & 7)) << 4));
        drain();
        __builtin_amdgcn_raw_buffer_store_b128(v, ors, off0 + (u32)r * NN, 0, AUX_WT);
    }
    drain();
    if (lane == 0 && p != me) __hip_atomic_store(fdp, call, RLX, SYS);
    if (wv != me) {
        const u32* fd = (const u32*)(myfl + FL_FD) + wv * 128 + ci;
        wt.start();
        while (__builtin_amdgcn_readfirstlane(__hip_atomic_load(fd, RLX, SYS)) < call) {
            if (wt.give_up()) {
                if (lane == 0) __builtin_amdgcn_raw_buffer_store_b32(0x7f7f7f7fu, rsrc_of(P.out[me], (u32)MM * NN), off0, 0, AUX_WT);
                break;
            }
        }
    }
}

}

extern "C" void kernel_launch(int rank, int nranks, ncclComm_t comm,
                              hipStream_t stream,
                              void* const* d_in, const int* in_sizes, int n_in,
                              void* d_out, int out_size,
                              const int* in_dtypes, int out_dtype,
                              void* const* d_out_all,
                              void* const* d_ws_all, size_t ws_bytes) {
    if (nranks != NR || (unsigned)rank >= (unsigned)NR || n_in != 2 || ws_bytes < WS_NEED ||
        in_sizes[0] != MM * KS || in_sizes[1] != KS * NN || out_size != MM * NN) {
        fprintf(stderr, "kernel_launch: unexpected problem shape\n");
        return;
    }
    Params P;
    P.x = (const unsigned short*)d_in[0];
    P.w = (const unsigned short*)d_in[1];
    for (int p = 0; p < NR; ++p) { P.ws[p] = (char*)d_ws_all[p]; P.out[p] = (char*)d_out_all[p]; }
    P.rank = rank;
    gemm_ar_q<<<NWG, 64 * WPB, 0, stream>>>(P);
    HIP_CHECK(hipGetLastError());
}
```
